```python
import numpy as np

kernel_name = "gpu_gemm_a2a_m8192_k8192_n8192_bf16_gelu"


ABSMAX_RTOL = 0.008

COMPUTE_DTYPE = "bfloat16"
EPILOGUE = "gelu"
IN_QUANT = None
OUT_QUANT = None
IN_DTYPES = {"x": "bfloat16", "w_mat": "bfloat16"}
WORLD_SIZE = 8

def _to_bf16(x):
    u = np.ascontiguousarray(x, dtype=np.float32).view(np.uint32)
    return ((u + 0x7FFF + ((u >> 16) & 1)) & 0xFFFF0000).view(np.float32)

def setup_inputs():

    x = _to_bf16(np.random.randn(8192, 8192).astype(np.float32))
    w_mat = _to_bf16((np.random.randn(8192, 8192) / np.sqrt(8192)).astype(np.float32))
    return {"x": x, "w_mat": w_mat}

def _epilogue(y):
    c = 0.7978845608028654
    return (0.5 * y * (1.0 + np.tanh(c * (y + 0.044715 * y ** 3)))).astype(np.float32)

def reference(x, w_mat):
    y = (x @ w_mat).astype(np.float32)
    return _epilogue(y)
```

```cpp
#define ADAPT 1
#define SPIN 300
#define OLDISSUE 1
#define DST 0x03010700
#define AUXPRIO 3
#define LATELD 1
#define SPDM 1
#define TRIG 0
#define PACE_N 1
#define PACE_D 1
#include <hip/hip_runtime.h>
#include <rccl/rccl.h>
#include <cstdio>
#include <utility>
typedef unsigned u32x4 __attribute__((ext_vector_type(4)));
typedef float f32x2 __attribute__((ext_vector_type(2)));
typedef float f32x4 __attribute__((ext_vector_type(4)));
typedef float f32x32 __attribute__((ext_vector_type(32)));
typedef __bf16 bf16x2 __attribute__((ext_vector_type(2)));
#define DI __device__ __forceinline__
#define WIN 0
#define STK 12
#define LDELAY 0
#define PRE 0
#define PRER NR
#define CPF 0
#define DST2 DST
#define KST 12
#define PSC "sc0 sc1"
#define PAUX 17
#define PSLEEP 2
#define RSTR 256
#define LEAD 0
#define DLY 1
#define SPLIT 0
#define PART16 1
struct Params { const char* x; const char* w; unsigned* myws; int rank; int pad; unsigned short* out[8]; unsigned* arr[8]; unsigned tiles[128]; };
constexpr unsigned long long DEADLINE_TICKS = 50000000ull;
constexpr int NR = 2, SLAST = 4, SKEW = 0;
constexpr unsigned long long RCFG = 0x10a000aull;
static const unsigned TILES[128] = {0x41b1a09u,0x51572635u,0x13120107u,0x335a315fu,0x56152403u,0xa111706u,0x21271e0du,0x1f0e2d2cu,0xb52192fu,0x1d1c232au,0x3f365d5cu,0xf160514u,0x3259375eu,0x2e55542bu,0x345b2229u,0x250c5302u,0x7c7b7a79u,0x68705878u,0x6c6b6a69u,0x4c4b4a49u,0x397f7e7du,0x40485060u,0x3b6f6e6du,0x3d4f4e4du,0x74737271u,0x28303818u,0x64636261u,0x44434241u,0x3a777675u,0x82010u,0x3c676665u,0x3e474645u,0x41b1a09u,0x51572635u,0x13120107u,0x335a315fu,0x56152403u,0xa111706u,0x21271e0du,0x1f0e2d2cu,0xb52192fu,0x1d1c232au,0x3f365d5cu,0xf160514u,0x3259375eu,0x2e55542bu,0x345b2229u,0x250c5302u,0x7c7b7a79u,0x68705878u,0x6c6b6a69u,0x4c4b4a49u,0x3b7f7e7du,0x40485060u,0x3d6f6e6du,0x394f4e4du,0x74737271u,0x28303818u,0x64636261u,0x44434241u,0x3c777675u,0x82010u,0x3e676665u,0x3a474645u,0x41b1a09u,0x51572635u,0x13120107u,0x335a315fu,0x56152403u,0xa111706u,0x21271e0du,0x1f0e2d2cu,0xb52192fu,0x1d1c232au,0x3f365d5cu,0xf160514u,0x3259375eu,0x2e55542bu,0x345b2229u,0x250c5302u,0x7c7b7a79u,0x68705878u,0x6c6b6a69u,0x4c4b4a49u,0x3d7f7e7du,0x40485060u,0x396f6e6du,0x3b4f4e4du,0x74737271u,0x28303818u,0x64636261u,0x44434241u,0x3e777675u,0x82010u,0x3a676665u,0x3c474645u,0x41b1a09u,0x51572635u,0x13120107u,0x335a315fu,0x56152403u,0xa111706u,0x21271e0du,0x1f0e2d2cu,0xb52192fu,0x1d1c232au,0x3f365d5cu,0xf160514u,0x3259375eu,0x2e55542bu,0x345b2229u,0x250c5302u,0x7c7b7a79u,0x68705878u,0x6c6b6a69u,0x4c4b4a49u,0x397f7e7du,0x40485060u,0x3b6f6e6du,0x3d4f4e4du,0x74737271u,0x28303818u,0x64636261u,0x44434241u,0x3a777675u,0x82010u,0x3c676665u,0x3e474645u};
constexpr unsigned W_TMO = 32;
constexpr unsigned W_ARR = 64;
constexpr unsigned W_SPD = 512;
constexpr unsigned W_CALL = 1024;
constexpr unsigned W_RNG = 9216;
constexpr unsigned W_FLAG = 16384;
constexpr unsigned W_DIAG = 16384 + 4*256*8*16;
constexpr size_t PART_OFF = 1u << 20;
constexpr unsigned PMT = PART16 ? 2048 : 4096, PWAVE = 8 * PMT, PBLK = 8 * PWAVE;
constexpr int LDS_BYTES = 5 * 32768;
constexpr int LM = SLAST == 8 ? 0 : 1;

DI float gelu1(float y) {
  const float k1 = -2.0f * 0.7978845608028654f * 1.4426950408889634f, k2 = k1 * 0.044715f;
  float t = y * __builtin_fmaf(y * y, k2, k1);
  float e = __builtin_amdgcn_exp2f(t);
  return y * __builtin_amdgcn_rcpf(1.0f + e);
}
DI f32x2 gelu2(f32x2 y) {
  const float k1 = -2.0f * 0.7978845608028654f * 1.4426950408889634f, k2 = k1 * 0.044715f;
  const f32x2 t = y * __builtin_elementwise_fma(y * y, (f32x2){k2, k2}, (f32x2){k1, k1});
  const f32x2 d = (f32x2){__builtin_amdgcn_exp2f(t.x), __builtin_amdgcn_exp2f(t.y)} + 1.0f;
  return y * (f32x2){__builtin_amdgcn_rcpf(d.x), __builtin_amdgcn_rcpf(d.y)};
}
DI unsigned pk2(f32x2 v) { bf16x2 b = __builtin_convertvector(v, bf16x2); return __builtin_bit_cast(unsigned, b); }
DI f32x2 up2(unsigned u) { return (f32x2){__uint_as_float(u << 16), __uint_as_float(u & 0xffff0000u)}; }
DI unsigned pk(float lo, float hi) { f32x2 v = {lo, hi}; bf16x2 b = __builtin_convertvector(v, bf16x2); return __builtin_bit_cast(unsigned, b); }
DI float lo16(unsigned u) { return __uint_as_float(u << 16); }
DI float hi16(unsigned u) { return __uint_as_float(u & 0xffff0000u); }
DI unsigned lane_id() { unsigned ln; asm volatile("v_mbcnt_lo_u32_b32 %0, -1, 0\n\tv_mbcnt_hi_u32_b32 %0, -1, %0" : "=v"(ln)); return ln; }
DI void store_wt(u32x4 v, unsigned voff, __amdgpu_buffer_rsrc_t rs, unsigned soff) {
  asm volatile("buffer_store_dwordx4 %0, %1, %2, %3 offen sc0 sc1\n\ts_nop 1" :: "v"(v), "v"(voff), "s"(rs), "s"(soff) : "memory");
}
DI void store_part(u32x4 v, unsigned voff, __amdgpu_buffer_rsrc_t rs, unsigned soff) {
  asm volatile("buffer_store_dwordx4 %0, %1, %2, %3 offen " PSC "\n\ts_nop 1" :: "v"(v), "v"(voff), "s"(rs), "s"(soff) : "memory");
}
DI void act_store(const float (&y)[8], unsigned voff, __amdgpu_buffer_rsrc_t rs, unsigned soff, bool bad) {
  unsigned x0 = pk(gelu1(y[0]), gelu1(y[1])), x1 = pk(gelu1(y[2]), gelu1(y[3])), y0 = pk(gelu1(y[4]), gelu1(y[5])), y1 = pk(gelu1(y[6]), gelu1(y[7]));
  auto r0 = __builtin_amdgcn_permlane16_swap(x0, y0, false, false);
  auto r1 = __builtin_amdgcn_permlane16_swap(x1, y1, false, false);
  u32x4 v = {r0[0], r1[0], r0[1], r1[1]};
  if (bad) v[0] = 0x7fc07fc0u;
  store_wt(v, voff, rs, soff);
}
template <int H> DI void dump_mt(const f32x32& a, unsigned voff, __amdgpu_buffer_rsrc_t rs, unsigned soff) {
  if constexpr (PART16) {
    #pragma unroll
    for (int i = 0; i < 2; ++i) { u32x4 v = {pk(a[16*H+8*i], a[16*H+8*i+1]), pk(a[16*H+8*i+2], a[16*H+8*i+3]), pk(a[16*H+8*i+4], a[16*H+8*i+5]), pk(a[16*H+8*i+6], a[16*H+8*i+7])};
      store_part(v, voff, rs, soff + i * 1024); }
  } else {
    #pragma unroll
    for (int i = 0; i < 4; ++i) { f32x4 v = {a[16*H+4*i], a[16*H+4*i+1], a[16*H+4*i+2], a[16*H+4*i+3]}; store_wt(__builtin_bit_cast(u32x4, v), voff, rs, soff + i * 1024); }
  }
}
DI void final_slice(const f32x32& a, const unsigned (&so)[3], unsigned lo, __amdgpu_buffer_rsrc_t rsP, unsigned voff, __amdgpu_buffer_rsrc_t rsO, unsigned soff, bool bad, bool remote) {
  u32x4 p[3][4];
  #pragma unroll
  for (int i = 0; i < 4; ++i)
    #pragma unroll
    for (int j = 0; j < 3; ++j) p[j][i] = __builtin_amdgcn_raw_buffer_load_b128(rsP, lo + i * 1024, so[j], PAUX);
  f32x2 y[4][4];
  #pragma unroll
  for (int i = 0; i < 4; ++i) {
    #pragma unroll
    for (int e = 0; e < 4; ++e) y[i][e] = ((f32x2){a[8 * i + 2 * e], a[8 * i + 2 * e + 1]} + up2(p[0][i][e])) + (up2(p[1][i][e]) + up2(p[2][i][e]));
    asm volatile("" : "+v"(y[i][0]), "+v"(y[i][1]), "+v"(y[i][2]), "+v"(y[i][3])); }
  #pragma unroll
  for (int i = 0; i < 4; ++i) {
    auto r0 = __builtin_amdgcn_permlane16_swap(pk2(gelu2(y[i][0])), pk2(gelu2(y[i][2])), false, false);
    auto r1 = __builtin_amdgcn_permlane16_swap(pk2(gelu2(y[i][1])), pk2(gelu2(y[i][3])), false, false);
    u32x4 v = {r0[0] | (bad ? 0x7fc07fc0u : 0u), r1[0], r0[1], r1[1]};
    if (i && remote) asm volatile("s_waitcnt vmcnt(%c1)" : "+v"(v) : "i"(WIN) : "memory");
    store_wt(v, voff, rsO, soff + (i >> 1) * 32768 + (i & 1) * 64); }
}
DI bool past_deadline(unsigned long long t0, unsigned* tmo, unsigned call) {
  if (__hip_atomic_load(tmo, __ATOMIC_RELAXED, __HIP_MEMORY_SCOPE_AGENT) == call || __builtin_amdgcn_s_memrealtime() - t0 > DEADLINE_TICKS) {
    __hip_atomic_store(tmo, call, __ATOMIC_RELAXED, __HIP_MEMORY_SCOPE_AGENT); return true; }
  return false;
}
DI bool wait_all_ge(unsigned* flag, unsigned need, unsigned* tmo, unsigned call) {
  const unsigned long long t0 = __builtin_amdgcn_s_memrealtime();
  for (unsigned spins = 0;; ++spins) {
    if (__all((int)(__hip_atomic_load(flag, __ATOMIC_RELAXED, __HIP_MEMORY_SCOPE_SYSTEM) - need) >= 0)) return false;
    if ((spins & 15) == 15 && __any(past_deadline(t0, tmo, call))) return true;
    __builtin_amdgcn_s_sleep(1);
  }
}
#define STAMP(i) do {} while (0)
struct Round { int ls, lx, slice, k0, nk; unsigned tw, pbase; };
DI int take(int i, int c) { return (int)((((c & 4) ? DST2 : DST) >> (8 * ((i + c) & 3))) & 255); }
DI int stag(int i, int c) { return take(i, c) - take(i + 1, c); }
DI Round round_cfg(const Params& p, int r, int b) {
  const int xcd = b & 7, c = b >> 5;
  Round R; R.ls = 2; R.lx = r; R.slice = xcd & 3;
  R.tw = p.tiles[((b >> 3) & 3) * 32 + r * 16 + (xcd >> 2) * 8 + (r ? (c & 4) + ((xcd + c) & 3) : c)];
  R.nk = 32; R.k0 = R.slice * 32;
  if (r == 0) { R.nk += stag(xcd, c); R.k0 -= take(xcd, c); }
  R.pbase = b;
  return R;
}
constexpr int LXL = 2, NKL = 128 / SLAST;
DI bool adapt_split(unsigned v, unsigned call, int xcd, int c, Round& R) {
  if (!__all((v >> 16) == (call & 0xffffu))) return false;
  constexpr int NX = 1 << LXL;
  const int g0 = xcd & ~(NX - 1);
  unsigned sp[NX], sum = 0;
  #pragma unroll
  for (int i = 0; i < NX; ++i) { const unsigned d = __builtin_amdgcn_readlane(v, g0 + i) & 0xffffu; sp[i] = (1u << 28) / (d < 1024u ? 1024u : d); sum += sp[i]; }
  { const unsigned mean = sum / NX, lo_ = mean - mean / 14, hi_ = mean + mean / 14; sum = 0;
    #pragma unroll
    for (int i = 0; i < NX; ++i) { sp[i] = sp[i] < lo_ ? lo_ : sp[i] > hi_ ? hi_ : sp[i]; sum += sp[i]; } }
  unsigned cum = 0; int lo = 0, hi = 0; const int prior = 32 + stag(xcd, c);
  #pragma unroll
  for (int i = 0; i < NX; ++i) {
    cum += sp[i];
    int e = (int)((cum * (unsigned)(128 + NX * prior) + sum / 2) / sum) - prior * (i + 1);
    if (i == NX - 1) e = NKL * NX;
    if (g0 + i == xcd - 1) lo = e;
    if (g0 + i == xcd) hi = e;
  }
  R.k0 = (R.slice >> LXL) * (NKL * NX) + lo; R.nk = hi - lo;
  return true;
}
DI void adapt_wait(unsigned* myws, unsigned call, int xcd, int c, Round& R) {
  const unsigned long long t0 = __builtin_amdgcn_s_memrealtime();
  for (;;) {
    const unsigned v = __hip_atomic_load(myws + W_SPD + 16 * (lane_id() & 7), __ATOMIC_RELAXED, __HIP_MEMORY_SCOPE_SYSTEM);
    if (adapt_split(v, call, xcd, c, R)) return;
    if (__any(past_deadline(t0, myws + W_TMO, call))) return;
    __builtin_amdgcn_s_sleep(1);
  }
}
DI int rng_nk(unsigned w) { const int n = (int)(w >> 8); return n < 2 ? 2 : n > 2 * NKL ? 2 * NKL : n; }
DI unsigned sload_now(const unsigned* q) {
  size_t a = (size_t)q; asm volatile("" : "+s"(a) :: "memory"); return *(const __attribute__((address_space(4))) unsigned*)a; }
DI unsigned part_blk(const Round& R, unsigned q) { const unsigned b = R.pbase; return R.lx ? (b & ~(3u | 96u)) | q | (((b + (b >> 5) - q) & 3) << 5) : (b & ~3u) | q; }

#define NOPN NOPS(NOPV)
#define NOPS(x) NOPT(x)
#define NOPT(x) #x
#define NOPV 0
#define ODD 2048
#define UNIT_A "s_mov_b32 m0, %0\n\ts_nop " NOPN "\n\tbuffer_load_dwordx4 %3, %1, %2 offen lds\n\ts_addk_i32 m0, 0x800\n\ts_nop " NOPN "\n\tbuffer_load_dwordx4 %4, %1, %2 offen lds\n\ts_addk_i32 m0, 0x800\n\ts_nop " NOPN "\n\tbuffer_load_dwordx4 %5, %1, %2 offen lds\n\ts_addk_i32 m0, 0x800\n\ts_nop " NOPN "\n\tbuffer_load_dwordx4 %6, %1, %2 offen lds\n\ts_addk_i32 m0, 0x800\n\ts_nop " NOPN "\n\tbuffer_load_dwordx4 %7, %1, %2 offen lds\n\ts_addk_i32 m0, 0x800\n\ts_nop " NOPN "\n\tbuffer_load_dwordx4 %8, %1, %2 offen lds\n\ts_addk_i32 m0, 0x800\n\ts_nop " NOPN "\n\tbuffer_load_dwordx4 %9, %1, %2 offen lds\n\ts_addk_i32 m0, 0x800\n\ts_nop " NOPN "\n\tbuffer_load_dwordx4 %10, %1, %2 offen lds\n\ts_addk_i32 m0, 0x800\n\ts_nop " NOPN "\n\tbuffer_load_dwordx4 %11, %1, %2 offen lds\n\ts_addk_i32 m0, 0x800\n\ts_nop " NOPN "\n\tbuffer_load_dwordx4 %12, %1, %2 offen lds\n\ts_addk_i32 m0, 0x800\n\ts_nop " NOPN "\n\tbuffer_load_dwordx4 %13, %1, %2 offen lds\n\ts_addk_i32 m0, 0x800\n\ts_nop " NOPN "\n\tbuffer_load_dwordx4 %14, %1, %2 offen lds\n\ts_addk_i32 m0, 0x800\n\ts_nop " NOPN "\n\tbuffer_load_dwordx4 %15, %1, %2 offen lds\n\ts_addk_i32 m0, 0x800\n\ts_nop " NOPN "\n\tbuffer_load_dwordx4 %16, %1, %2 offen lds\n\ts_addk_i32 m0, 0x800\n\ts_nop " NOPN "\n\tbuffer_load_dwordx4 %17, %1, %2 offen lds\n\ts_addk_i32 m0, 0x800\n\ts_nop " NOPN "\n\tbuffer_load_dwordx4 %18, %1, %2 offen lds"
#define UNIT_B "s_mov_b32 m0, %0\n\ts_nop " NOPN "\n\tbuffer_load_dwordx4 %2, %1, %6 offen lds\n\ts_addk_i32 m0, 0x800\n\ts_nop " NOPN "\n\tbuffer_load_dwordx4 %3, %1, %6 offen lds\n\ts_addk_i32 m0, 0x800\n\ts_nop " NOPN "\n\tbuffer_load_dwordx4 %4, %1, %6 offen lds\n\ts_addk_i32 m0, 0x800\n\ts_nop " NOPN "\n\tbuffer_load_dwordx4 %5, %1, %6 offen lds\n\ts_addk_i32 m0, 0x800\n\ts_nop " NOPN "\n\tbuffer_load_dwordx4 %2, %1, %7 offen lds\n\ts_addk_i32 m0, 0x800\n\ts_nop " NOPN "\n\tbuffer_load_dwordx4 %3, %1, %7 offen lds\n\ts_addk_i32 m0, 0x800\n\ts_nop " NOPN "\n\tbuffer_load_dwordx4 %4, %1, %7 offen lds\n\ts_addk_i32 m0, 0x800\n\ts_nop " NOPN "\n\tbuffer_load_dwordx4 %5, %1, %7 offen lds\n\ts_addk_i32 m0, 0x800\n\ts_nop " NOPN "\n\tbuffer_load_dwordx4 %2, %1, %8 offen lds\n\ts_addk_i32 m0, 0x800\n\ts_nop " NOPN "\n\tbuffer_load_dwordx4 %3, %1, %8 offen lds\n\ts_addk_i32 m0, 0x800\n\ts_nop " NOPN "\n\tbuffer_load_dwordx4 %4, %1, %8 offen lds\n\ts_addk_i32 m0, 0x800\n\ts_nop " NOPN "\n\tbuffer_load_dwordx4 %5, %1, %8 offen lds\n\ts_addk_i32 m0, 0x800\n\ts_nop " NOPN "\n\tbuffer_load_dwordx4 %2, %1, %9 offen lds\n\ts_addk_i32 m0, 0x800\n\ts_nop " NOPN "\n\tbuffer_load_dwordx4 %3, %1, %9 offen lds\n\ts_addk_i32 m0, 0x800\n\ts_nop " NOPN "\n\tbuffer_load_dwordx4 %4, %1, %9 offen lds\n\ts_addk_i32 m0, 0x800\n\ts_nop " NOPN "\n\tbuffer_load_dwordx4 %5, %1, %9 offen lds"
__global__ __launch_bounds__(768,3) void gemm_a2a(Params p) {
  extern __shared__ __attribute__((aligned(1024))) char smem[];
  const int wave = __builtin_amdgcn_readfirstlane(threadIdx.x >> 6);
  const int b = blockIdx.x, xcd = b & 7, m = (b >> 3) & 3, c = b >> 5;
  const int PRIOR = 64 - NKL + stag(xcd, c);
  int TOTAL = PRIOR + NKL;
  const unsigned rt_start = (unsigned)__builtin_amdgcn_s_memrealtime();
  const int rank = p.rank, row0 = m * 256;
  const unsigned ldsbase = (unsigned)(size_t)(__attribute__((address_space(3))) char*)smem;
  STAMP(0);
  size_t pc0 = 0;
  if (CPF) { asm volatile("s_getpc_b64 %0" : "=s"(pc0)); pc0 &= ~(size_t)255; }
  unsigned* myws = p.myws;
  auto load_call = [&]() { return sload_now(myws + W_CALL + 32 * b) + 1; };
  if (__builtin_expect(wave == 8 || wave == 9, 1)) {
    const int lw = wave & 1; const unsigned lane = lane_id();
    unsigned tw[NR];
    #pragma unroll
    for (int r = 0; r < NR; ++r) tw[r] = round_cfg(p, r, b).tw;
    const __amdgpu_buffer_rsrc_t srdA = __builtin_amdgcn_make_buffer_rsrc((void*)(p.x - 2048), (short)0, 1024*8192*2 + 2048, 0x00020000);
    const __amdgpu_buffer_rsrc_t srdB = __builtin_amdgcn_make_buffer_rsrc((void*)(p.w - 2048), (short)0, 8192*8192*2 + 2048, 0x00020000);
    const unsigned voffA = (lane>>3)*16384u + (((lane&7) ^ ((lw<<2) | (lane>>4)))*16u);
    const unsigned voffB = (lane>>3)*16384u + (((lane&7) ^ ((((lane>>4)&1) | (lw<<1))<<1))*16u);
    const unsigned vb0 = voffB + 2048, vb1 = voffB + 262144 + ODD, vb2 = voffB + 2 * 262144 + 2048, vb3 = voffB + 3 * 262144 + ODD;
    unsigned va[16];
    #pragma unroll
    for (int i = 0; i < 16; ++i) va[i] = voffA + (i << 18) + ((i & 1) ? ODD : 2048);
    int r = 0, left = 0, kst = 0, kt = 0, nk = 0, khi = 0, kj = 0;
    int ph = 1 + (c & 3);
    unsigned twc = 0, m0v = ldsbase + lw * 1024; int lk0 = 0;
    const int kx = m;
    #pragma unroll 1
    for (int t = -1; t <= TOTAL; ++t) {
      if (t >= 0 && t < TOTAL) {
        asm volatile(UNIT_A
          :: "s"(m0v), "s"(srdA), "s"((unsigned)((row0 + lw*8) * 8192 + kt*64) * 2u), "v"(va[0]), "v"(va[1]), "v"(va[2]), "v"(va[3]), "v"(va[4]), "v"(va[5]), "v"(va[6]), "v"(va[7]),
             "v"(va[8]), "v"(va[9]), "v"(va[10]), "v"(va[11]), "v"(va[12]), "v"(va[13]), "v"(va[14]), "v"(va[15]) : "scc", "memory");
        m0v = m0v >= ldsbase + 4 * 32768 ? m0v - 4 * 32768 : m0v + 32768;
      }
      if (t + 1 < TOTAL) {
        if (left == 0) {
          Round R = round_cfg(p, r, b); twc = r == 0 ? tw[0] : r == 1 ? tw[1] : tw[NR - 1];
          if (r == NR - 1) {
            if (ADAPT) { R.k0 = lk0; R.nk = TOTAL - PRIOR; }
            ph = 1 + ((xcd + c + (c >> 2) * 2) & 3);
            #pragma unroll
            for (int i = 0; i < 16; ++i) va[i] = voffA + ((unsigned)(i ^ (R.slice << LM)) << 18) + ((i & 1) ? ODD : 2048);
          }
          left = nk = R.nk; khi = R.k0 + R.nk; kj = R.k0 > R.slice * 32 ? R.k0 : R.slice * 32; kst = r == 0 ? 8 : 8 + PRIOR + DLY;
          ++r;
        }
        { const int j = (kj & 96) | ((kj + kst) & 31);
          kt = (r == 1 && (unsigned)((j & 31) - (4 + ph)) < 4u) ? j : (((j - ph) ^ kx) + ph) & 127; }
        ++kj; if (kj == khi) kj -= nk;
        --left;
        const unsigned sB = (unsigned)(kt*64 + lw*8) * 16384u;
        asm volatile(UNIT_B
          :: "s"(m0v), "s"(srdB), "v"(vb0), "v"(vb1), "v"(vb2), "v"(vb3),
             "s"(sB + (twc & 127) * 128), "s"(sB + ((twc >> 8) & 127) * 128), "s"(sB + ((twc >> 16) & 127) * 128), "s"(sB + (twc >> 24) * 128) : "scc", "memory");
        m0v = m0v >= ldsbase + 4 * 32768 ? m0v - 4 * 32768 : m0v + 32768;
        if (t >= 0) asm volatile("s_waitcnt vmcnt(16)\n\ts_barrier" ::: "memory");
      } else asm volatile("s_waitcnt vmcnt(0)\n\ts_barrier" ::: "memory");
      if (ADAPT && t == PRIOR - 3) { const unsigned w = sload_now(myws + W_RNG + 16 * b); lk0 = w & 127; TOTAL = PRIOR + rng_nk(w); }
    }
    return;
  }
  const __amdgpu_buffer_rsrc_t srdP = __builtin_amdgcn_make_buffer_rsrc((char*)myws + PART_OFF, (short)0, NR * 256 * PBLK, 0x00020000);
  if (wave >= 10) {
    const int a = wave - 10; const unsigned lane = lane_id();
    __builtin_amdgcn_s_setprio(AUXPRIO);
    int nb = 2;
    asm volatile("s_barrier\n\ts_barrier" ::: "memory");
    const unsigned rt2 = (unsigned)__builtin_amdgcn_s_memrealtime();
    const unsigned call = load_call();
    auto code_prefetch = [&]() {
      const __amdgpu_buffer_rsrc_t rc = __builtin_amdgcn_make_buffer_rsrc((void*)pc0, (short)0, CPF * 1024, 0x00020000);
      for (int i = a; i < CPF / 4; i += 2) asm volatile("buffer_load_dword v100, %0, %1, %2 offen" :: "v"(lane * 64u), "s"(rc), "s"(i * 4096) : "v100", "memory");
    };
    if (CPF) code_prefetch();
    if (ADAPT) {
      while (nb < STK + 1) { asm volatile("s_barrier" ::: "memory"); ++nb; }
      if (a == 0 && b < 8 && lane == 0) { unsigned d = (unsigned)__builtin_amdgcn_s_memrealtime() - (SPDM ? rt2 : rt_start); d = d > 0xffffu ? 0xffffu : d;
        __hip_atomic_store(myws + W_SPD + 16 * xcd, (call << 16) | d, __ATOMIC_RELAXED, __HIP_MEMORY_SCOPE_SYSTEM); }
      while (nb < STK + 6) { asm volatile("s_barrier" ::: "memory"); ++nb; }
      if (a == 0) { Round R = round_cfg(p, NR - 1, b); adapt_wait(myws, call, xcd, c, R);
        if (lane == 0) __hip_atomic_store(myws + W_RNG + 16 * b, (unsigned)(R.k0 | (R.nk << 8)), __ATOMIC_RELAXED, __HIP_MEMORY_SCOPE_SYSTEM);
        asm volatile("s_barrier\n\ts_waitcnt vmcnt(0)" ::: "memory"); ++nb; }
      while (nb < STK + 8) { asm volatile("s_barrier" ::: "memory"); ++nb; }
      TOTAL = PRIOR + rng_nk(sload_now(myws + W_RNG + 16 * b));
    }
    unsigned t_tick = 0;
    auto tick = [&]() { if (nb <= TOTAL) { asm volatile("s_barrier" ::: "memory"); ++nb; t_tick = (unsigned)__builtin_amdgcn_s_memtime(); } else __builtin_amdgcn_s_sleep(1); };
    auto spare = [&](unsigned budget) { return nb > TOTAL || (unsigned)__builtin_amdgcn_s_memtime() - t_tick < budget; };
    auto vm_pending = [&]() { unsigned v; asm volatile("s_getreg_b32 %0, hwreg(HW_REG_IB_STS, 0, 4)" : "=s"(v)); return v; };
    auto wait_vm = [&](unsigned n, unsigned budget) {
      while (nb <= TOTAL && vm_pending() > n) { if (spare(budget)) __builtin_amdgcn_s_sleep(1); else tick(); } };
    const unsigned voffO = (lane & 15) * 2048u + (lane & 16) * 2u + (lane & 32) / 2u, lo = lane * 16u;
    int kend = 0, sched = -1;
    #pragma unroll 1
    for (int r = 0; r + 1 < NR; ++r) {
      const Round R = round_cfg(p, r, b);
      kend += R.nk;
      const unsigned tw = R.tw >> (16 * a);
      const __amdgpu_buffer_rsrc_t srdO0 = __builtin_amdgcn_make_buffer_rsrc(p.out[(tw >> 4) & 7], (short)0, 8192 * 1024 * 2, 0x00020000);
      const __amdgpu_buffer_rsrc_t srdO1 = __builtin_amdgcn_make_buffer_rsrc(p.out[(tw >> 12) & 7], (short)0, 8192 * 1024 * 2, 0x00020000);
      if (CPF && r == 0) { while (nb < kend - 3) tick(); code_prefetch(); }
      while (nb < kend + TRIG + 2) tick();
      if (CPF && r == NR - 2) code_prefetch();
      bool bad = false;
      { const unsigned fo = (W_FLAG + 16u * ((r * 256 + part_blk(R, (lane >> 2) & ((1 << R.ls) - 1))) * 8 + (lane & 2) * 2 + 2 * a + (lane & 1))) * 4u;
        const unsigned long long t0 = __builtin_amdgcn_s_memrealtime();
        for (;;) {
          unsigned f;
          asm volatile("global_load_dword v100, %0, %1 sc0 sc1" :: "v"(fo), "s"(myws) : "v100", "memory");
          wait_vm(0, 1700);
          asm volatile("s_waitcnt vmcnt(0)\n\tv_mov_b32 %0, v100" : "=v"(f) :: "memory");
          if (__all((int)(f - call) >= 0)) break;
          if (!spare(1700)) tick();
          if (nb > TOTAL && __any(past_deadline(t0, myws + W_TMO, call))) { bad = true; break; }
        } }
      if (r == 0) STAMP(1); else STAMP(3);
      const unsigned badm = bad ? 0x7fc07fc0u : 0u;
      const int lm = 3 - R.ls, nch = 8 << lm;
#define LD1(R0, R1, Q) asm volatile("buffer_load_dwordx4 v[" #R0 ":" #R1 "], %0, %1, %2 offen " PSC :: "v"(lo), "s"(srdP), "s"(so + part_blk(R, Q) * PBLK) : "memory")
      auto chunk_src = [&](int n) { return r * RSTR * PBLK + (((n >> 2) & 1) * 4 + 2 * a + (n & 1)) * PWAVE + ((R.slice << lm) + (n >> 3)) * PMT + ((n >> 1) & 1) * (PMT / 2); };
      auto loads0 = [&](int n) { const unsigned so = chunk_src(n);
        LD1(68, 71, 0); LD1(72, 75, 1); LD1(76, 79, 2); LD1(80, 83, 3); if (R.ls == 3) { LD1(84, 87, 4); LD1(88, 91, 5); LD1(92, 95, 6); LD1(96, 99, 7); } };
      auto loads1 = [&](int n) { const unsigned so = chunk_src(n);
        LD1(100, 103, 0); LD1(104, 107, 1); LD1(108, 111, 2); LD1(112, 115, 3); if (R.ls == 3) { LD1(116, 119, 4); LD1(120, 123, 5); LD1(124, 127, 6); LD1(128, 131, 7); } };
      loads0(0); loads1(1);
      #pragma unroll 1
      for (int n = 0; n < nch; ++n) {
        if (sched < 0) sched = nb * PACE_N;
        while (nb <= TOTAL && nb * PACE_N < sched) tick();
        sched += PACE_D;
        const unsigned budget = R.ls == 3 ? SPIN : SPIN + 500;
        const unsigned young = (n + 1 < nch ? 1u << R.ls : 0u) + (n >= 1 ? 1 : 0);
        wait_vm(young, budget); if (!spare(budget)) tick();
        switch (young) {
          case 1: asm volatile("s_waitcnt vmcnt(1)" ::: "memory"); break;
          case 2: asm volatile("s_waitcnt vmcnt(2)" ::: "memory"); break;
          case 4: asm volatile("s_waitcnt vmcnt(4)" ::: "memory"); break;
          case 5: asm volatile("s_waitcnt vmcnt(5)" ::: "memory"); break;
          case 6: asm volatile("s_waitcnt vmcnt(6)" ::: "memory"); break;
          case 8: asm volatile("s_waitcnt vmcnt(8)" ::: "memory"); break;
          case 9: asm volatile("s_waitcnt vmcnt(9)" ::: "memory"); break;
          case 10: asm volatile("s_waitcnt vmcnt(10)" ::: "memory"); break;
          default: asm volatile("s_waitcnt vmcnt(0)" ::: "memory"); break;
        }
        if (n & 1) {
          asm volatile(
          "v_lshlrev_b32 v132, 16, v100\n"
          "v_and_b32 v133, 0xffff0000, v100\n"
          "s_nop 0\n"
          "v_lshlrev_b32 v140, 16, v104\n"
          "v_and_b32 v141, 0xffff0000, v104\n"
          "s_nop 0\n"
          "v_add_f32 v132, v132, v140\n"
          "v_add_f32 v133, v133, v141\n"
          "s_nop 0\n"
          "v_lshlrev_b32 v140, 16, v108\n"
          "v_and_b32 v141, 0xffff0000, v108\n"
          "s_nop 0\n"
          "v_add_f32 v132, v132, v140\n"
          "v_add_f32 v133, v133, v141\n"
          "s_nop 0\n"
          "v_lshlrev_b32 v140, 16, v112\n"
          "v_and_b32 v141, 0xffff0000, v112\n"
          "s_nop 0\n"
          "v_add_f32 v132, v132, v140\n"
          "v_add_f32 v133, v133, v141\n"
          "s_nop 0\n"
          "v_lshlrev_b32 v134, 16, v101\n"
          "v_and_b32 v135, 0xffff0000, v101\n"
          "s_nop 0\n"
          "v_lshlrev_b32 v140, 16, v105\n"
          "v_and_b32 v141, 0xffff0000, v105\n"
          "s_nop 0\n"
          "v_add_f32 v134, v134, v140\n"
          "v_add_f32 v135, v135, v141\n"
          "s_nop 0\n"
          "v_lshlrev_b32 v140, 16, v109\n"
          "v_and_b32 v141, 0xffff0000, v109\n"
          "s_nop 0\n"
          "v_add_f32 v134, v134, v140\n"
          "v_add_f32 v135, v135, v141\n"
          "s_nop 0\n"
          "v_lshlrev_b32 v140, 16, v113\n"
          "v_and_b32 v141, 0xffff0000, v113\n"
          "s_nop 0\n"
          "v_add_f32 v134, v134, v140\n"
          "v_add_f32 v135, v135, v141\n"
          "s_nop 0\n"
          "v_lshlrev_b32 v136, 16, v102\n"
          "v_and_b32 v137, 0xffff0000, v102\n"
          "s_nop 0\n"
          "v_lshlrev_b32 v140, 16, v106\n"
          "v_and_b32 v141, 0xffff0000, v106\n"
          "s_nop 0\n"
          "v_add_f32 v136, v136, v140\n"
          "v_add_f32 v137, v137, v141\n"
          "s_nop 0\n"
          "v_lshlrev_b32 v140, 16, v110\n"
          "v_and_b32 v141, 0xffff0000, v110\n"
          "s_nop 0\n"
          "v_add_f32 v136, v136, v140\n"
          "v_add_f32 v137, v137, v141\n"
          "s_nop 0\n"
          "v_lshlrev_b32 v140, 16, v114\n"
          "v_and_b32 v141, 0xffff0000, v114\n"
          "s_nop 0\n"
          "v_add_f32 v136, v136, v140\n"
          "v_add_f32 v137, v137, v141\n"
          "s_nop 0\n"
          "v_lshlrev_b32 v138, 16, v103\n"
          "v_and_b32 v139, 0xffff0000, v103\n"
          "s_nop 0\n"
          "v_lshlrev_b32 v140, 16, v107\n"
          "v_and_b32 v141, 0xffff0000, v107\n"
          "s_nop 0\n"
          "v_add_f32 v138, v138, v140\n"
          "v_add_f32 v139, v139, v141\n"
          "s_nop 0\n"
          "v_lshlrev_b32 v140, 16, v111\n"
          "v_and_b32 v141, 0xffff0000, v111\n"
          "s_nop 0\n"
          "v_add_f32 v138, v138, v140\n"
          "v_add_f32 v139, v139, v141\n"
          "s_nop 0\n"
          "v_lshlrev_b32 v140, 16, v115\n"
          "v_and_b32 v141, 0xffff0000, v115\n"
          "s_nop 0\n"
          "v_add_f32 v138, v138, v140\n"
          "v_add_f32 v139, v139, v141\n"
          "s_nop 0\n"
          ::: "v68","v69","v70","v71","v72","v73","v74","v75","v76","v77","v78","v79","v80","v81","v82","v83","v84","v85","v86","v87","v88","v89","v90","v91","v92","v93","v94","v95","v96","v97","v98","v99","v100","v101","v102","v103","v104","v105","v106","v107","v108","v109","v110","v111","v112","v113","v114","v115","v116","v117","v118","v119","v120","v121","v122","v123","v124","v125","v126","v127","v128","v129","v130","v131","v132","v133","v134","v135","v136","v137","v138","v139","v140","v141","v142","v143","v144","v145","v146","v147","v148","v149","v150","v151","v152", "memory");
          if (R.ls == 3) asm volatile(
          "v_lshlrev_b32 v140, 16, v116\n"
          "v_and_b32 v141, 0xffff0000, v116\n"
          "s_nop 0\n"
          "v_add_f32 v132, v132, v140\n"
          "v_add_f32 v133, v133, v141\n"
          "s_nop 0\n"
          "v_lshlrev_b32 v140, 16, v120\n"
          "v_and_b32 v141, 0xffff0000, v120\n"
          "s_nop 0\n"
          "v_add_f32 v132, v132, v140\n"
          "v_add_f32 v133, v133, v141\n"
          "s_nop 0\n"
          "v_lshlrev_b32 v140, 16, v124\n"
          "v_and_b32 v141, 0xffff0000, v124\n"
          "s_nop 0\n"
          "v_add_f32 v132, v132, v140\n"
          "v_add_f32 v133, v133, v141\n"
          "s_nop 0\n"
          "v_lshlrev_b32 v140, 16, v128\n"
          "v_and_b32 v141, 0xffff0000, v128\n"
          "s_nop 0\n"
          "v_add_f32 v132, v132, v140\n"
          "v_add_f32 v133, v133, v141\n"
          "s_nop 0\n"
          "v_lshlrev_b32 v140, 16, v117\n"
          "v_and_b32 v141, 0xffff0000, v117\n"
          "s_nop 0\n"
          "v_add_f32 v134, v134, v140\n"
          "v_add_f32 v135, v135, v141\n"
          "s_nop 0\n"
          "v_lshlrev_b32 v140, 16, v121\n"
          "v_and_b32 v141, 0xffff0000, v121\n"
          "s_nop 0\n"
          "v_add_f32 v134, v134, v140\n"
          "v_add_f32 v135, v135, v141\n"
          "s_nop 0\n"
          "v_lshlrev_b32 v140, 16, v125\n"
          "v_and_b32 v141, 0xffff0000, v125\n"
          "s_nop 0\n"
          "v_add_f32 v134, v134, v140\n"
          "v_add_f32 v135, v135, v141\n"
          "s_nop 0\n"
          "v_lshlrev_b32 v140, 16, v129\n"
          "v_and_b32 v141, 0xffff0000, v129\n"
          "s_nop 0\n"
          "v_add_f32 v134, v134, v140\n"
          "v_add_f32 v135, v135, v141\n"
          "s_nop 0\n"
          "v_lshlrev_b32 v140, 16, v118\n"
          "v_and_b32 v141, 0xffff0000, v118\n"
          "s_nop 0\n"
          "v_add_f32 v136, v136, v140\n"
          "v_add_f32 v137, v137, v141\n"
          "s_nop 0\n"
          "v_lshlrev_b32 v140, 16, v122\n"
          "v_and_b32 v141, 0xffff0000, v122\n"
          "s_nop 0\n"
          "v_add_f32 v136, v136, v140\n"
          "v_add_f32 v137, v137, v141\n"
          "s_nop 0\n"
          "v_lshlrev_b32 v140, 16, v126\n"
          "v_and_b32 v141, 0xffff0000, v126\n"
          "s_nop 0\n"
          "v_add_f32 v136, v136, v140\n"
          "v_add_f32 v137, v137, v141\n"
          "s_nop 0\n"
          "v_lshlrev_b32 v140, 16, v130\n"
          "v_and_b32 v141, 0xffff0000, v130\n"
          "s_nop 0\n"
          "v_add_f32 v136, v136, v140\n"
          "v_add_f32 v137, v137, v141\n"
          "s_nop 0\n"
          "v_lshlrev_b32 v140, 16, v119\n"
          "v_and_b32 v141, 0xffff0000, v119\n"
          "s_nop 0\n"
          "v_add_f32 v138, v138, v140\n"
          "v_add_f32 v139, v139, v141\n"
          "s_nop 0\n"
          "v_lshlrev_b32 v140, 16, v123\n"
          "v_and_b32 v141, 0xffff0000, v123\n"
          "s_nop 0\n"
          "v_add_f32 v138, v138, v140\n"
          "v_add_f32 v139, v139, v141\n"
          "s_nop 0\n"
          "v_lshlrev_b32 v140, 16, v127\n"
          "v_and_b32 v141, 0xffff0000, v127\n"
          "s_nop 0\n"
          "v_add_f32 v138, v138, v140\n"
          "v_add_f32 v139, v139, v141\n"
          "s_nop 0\n"
          "v_lshlrev_b32 v140, 16, v131\n"
          "v_and_b32 v141, 0xffff0000, v131\n"
          "s_nop 0\n"
          "v_add_f32 v138, v138, v140\n"
          "v_add_f32 v139, v139, v141\n"
          "s_nop 0\n"
          ::: "v68","v69","v70","v71","v72","v73","v74","v75","v76","v77","v78","v79","v80","v81","v82","v83","v84","v85","v86","v87","v88","v89","v90","v91","v92","v93","v94","v95","v96","v97","v98","v99","v100","v101","v102","v103","v104","v105","v106","v107","v108","v109","v110","v111","v112","v113","v114","v115","v116","v117","v118","v119","v120","v121","v122","v123","v124","v125","v126","v127","v128","v129","v130","v131","v132","v133","v134","v135","v136","v137","v138","v139","v140","v141","v142","v143","v144","v145","v146","v147","v148","v149","v150","v151","v152", "memory");
          if (!LATELD && n + 2 < nch) loads1(n + 2);
        } else {
          asm volatile(
          "v_lshlrev_b32 v132, 16, v68\n"
          "v_and_b32 v133, 0xffff0000, v68\n"
          "s_nop 0\n"
          "v_lshlrev_b32 v140, 16, v72\n"
          "v_and_b32 v141, 0xffff0000, v72\n"
          "s_nop 0\n"
          "v_add_f32 v132, v132, v140\n"
          "v_add_f32 v133, v133, v141\n"
          "s_nop 0\n"
          "v_lshlrev_b32 v140, 16, v76\n"
          "v_and_b32 v141, 0xffff0000, v76\n"
          "s_nop 0\n"
          "v_add_f32 v132, v132, v140\n"
          "v_add_f32 v133, v133, v141\n"
          "s_nop 0\n"
          "v_lshlrev_b32 v140, 16, v80\n"
          "v_and_b32 v141, 0xffff0000, v80\n"
          "s_nop 0\n"
          "v_add_f32 v132, v132, v140\n"
          "v_add_f32 v133, v133, v141\n"
          "s_nop 0\n"
          "v_lshlrev_b32 v134, 16, v69\n"
          "v_and_b32 v135, 0xffff0000, v69\n"
          "s_nop 0\n"
          "v_lshlrev_b32 v140, 16, v73\n"
          "v_and_b32 v141, 0xffff0000, v73\n"
          "s_nop 0\n"
          "v_add_f32 v134, v134, v140\n"
          "v_add_f32 v135, v135, v141\n"
          "s_nop 0\n"
          "v_lshlrev_b32 v140, 16, v77\n"
          "v_and_b32 v141, 0xffff0000, v77\n"
          "s_nop 0\n"
          "v_add_f32 v134, v134, v140\n"
          "v_add_f32 v135, v135, v141\n"
          "s_nop 0\n"
          "v_lshlrev_b32 v140, 16, v81\n"
          "v_and_b32 v141, 0xffff0000, v81\n"
          "s_nop 0\n"
          "v_add_f32 v134, v134, v140\n"
          "v_add_f32 v135, v135, v141\n"
          "s_nop 0\n"
          "v_lshlrev_b32 v136, 16, v70\n"
          "v_and_b32 v137, 0xffff0000, v70\n"
          "s_nop 0\n"
          "v_lshlrev_b32 v140, 16, v74\n"
          "v_and_b32 v141, 0xffff0000, v74\n"
          "s_nop 0\n"
          "v_add_f32 v136, v136, v140\n"
          "v_add_f32 v137, v137, v141\n"
          "s_nop 0\n"
          "v_lshlrev_b32 v140, 16, v78\n"
          "v_and_b32 v141, 0xffff0000, v78\n"
          "s_nop 0\n"
          "v_add_f32 v136, v136, v140\n"
          "v_add_f32 v137, v137, v141\n"
          "s_nop 0\n"
          "v_lshlrev_b32 v140, 16, v82\n"
          "v_and_b32 v141, 0xffff0000, v82\n"
          "s_nop 0\n"
          "v_add_f32 v136, v136, v140\n"
          "v_add_f32 v137, v137, v141\n"
          "s_nop 0\n"
          "v_lshlrev_b32 v138, 16, v71\n"
          "v_and_b32 v139, 0xffff0000, v71\n"
          "s_nop 0\n"
          "v_lshlrev_b32 v140, 16, v75\n"
          "v_and_b32 v141, 0xffff0000, v75\n"
          "s_nop 0\n"
          "v_add_f32 v138, v138, v140\n"
          "v_add_f32 v139, v139, v141\n"
          "s_nop 0\n"
          "v_lshlrev_b32 v140, 16, v79\n"
          "v_and_b32 v141, 0xffff0000, v79\n"
          "s_nop 0\n"
          "v_add_f32 v138, v138, v140\n"
          "v_add_f32 v139, v139, v141\n"
          "s_nop 0\n"
          "v_lshlrev_b32 v140, 16, v83\n"
          "v_and_b32 v141, 0xffff0000, v83\n"
          "s_nop 0\n"
          "v_add_f32 v138, v138, v140\n"
          "v_add_f32 v139, v139, v141\n"
          "s_nop 0\n"
          ::: "v68","v69","v70","v71","v72","v73","v74","v75","v76","v77","v78","v79","v80","v81","v82","v83","v84","v85","v86","v87","v88","v89","v90","v91","v92","v93","v94","v95","v96","v97","v98","v99","v100","v101","v102","v103","v104","v105","v106","v107","v108","v109","v110","v111","v112","v113","v114","v115","v116","v117","v118","v119","v120","v121","v122","v123","v124","v125","v126","v127","v128","v129","v130","v131","v132","v133","v134","v135","v136","v137","v138","v139","v140","v141","v142","v143","v144","v145","v146","v147","v148","v149","v150","v151","v152", "memory");
          if (R.ls == 3) asm volatile(
          "v_lshlrev_b32 v140, 16, v84\n"
          "v_and_b32 v141, 0xffff0000, v84\n"
          "s_nop 0\n"
          "v_add_f32 v132, v132, v140\n"
          "v_add_f32 v133, v133, v141\n"
          "s_nop 0\n"
          "v_lshlrev_b32 v140, 16, v88\n"
          "v_and_b32 v141, 0xffff0000, v88\n"
          "s_nop 0\n"
          "v_add_f32 v132, v132, v140\n"
          "v_add_f32 v133, v133, v141\n"
          "s_nop 0\n"
          "v_lshlrev_b32 v140, 16, v92\n"
          "v_and_b32 v141, 0xffff0000, v92\n"
          "s_nop 0\n"
          "v_add_f32 v132, v132, v140\n"
          "v_add_f32 v133, v133, v141\n"
          "s_nop 0\n"
          "v_lshlrev_b32 v140, 16, v96\n"
          "v_and_b32 v141, 0xffff0000, v96\n"
          "s_nop 0\n"
          "v_add_f32 v132, v132, v140\n"
          "v_add_f32 v133, v133, v141\n"
          "s_nop 0\n"
          "v_lshlrev_b32 v140, 16, v85\n"
          "v_and_b32 v141, 0xffff0000, v85\n"
          "s_nop 0\n"
          "v_add_f32 v134, v134, v140\n"
          "v_add_f32 v135, v135, v141\n"
          "s_nop 0\n"
          "v_lshlrev_b32 v140, 16, v89\n"
          "v_and_b32 v141, 0xffff0000, v89\n"
          "s_nop 0\n"
          "v_add_f32 v134, v134, v140\n"
          "v_add_f32 v135, v135, v141\n"
          "s_nop 0\n"
          "v_lshlrev_b32 v140, 16, v93\n"
          "v_and_b32 v141, 0xffff0000, v93\n"
          "s_nop 0\n"
          "v_add_f32 v134, v134, v140\n"
          "v_add_f32 v135, v135, v141\n"
          "s_nop 0\n"
          "v_lshlrev_b32 v140, 16, v97\n"
          "v_and_b32 v141, 0xffff0000, v97\n"
          "s_nop 0\n"
          "v_add_f32 v134, v134, v140\n"
          "v_add_f32 v135, v135, v141\n"
          "s_nop 0\n"
          "v_lshlrev_b32 v140, 16, v86\n"
          "v_and_b32 v141, 0xffff0000, v86\n"
          "s_nop 0\n"
          "v_add_f32 v136, v136, v140\n"
          "v_add_f32 v137, v137, v141\n"
          "s_nop 0\n"
          "v_lshlrev_b32 v140, 16, v90\n"
          "v_and_b32 v141, 0xffff0000, v90\n"
          "s_nop 0\n"
          "v_add_f32 v136, v136, v140\n"
          "v_add_f32 v137, v137, v141\n"
          "s_nop 0\n"
          "v_lshlrev_b32 v140, 16, v94\n"
          "v_and_b32 v141, 0xffff0000, v94\n"
          "s_nop 0\n"
          "v_add_f32 v136, v136, v140\n"
          "v_add_f32 v137, v137, v141\n"
          "s_nop 0\n"
          "v_lshlrev_b32 v140, 16, v98\n"
          "v_and_b32 v141, 0xffff0000, v98\n"
          "s_nop 0\n"
          "v_add_f32 v136, v136, v140\n"
          "v_add_f32 v137, v137, v141\n"
          "s_nop 0\n"
          "v_lshlrev_b32 v140, 16, v87\n"
          "v_and_b32 v141, 0xffff0000, v87\n"
          "s_nop 0\n"
          "v_add_f32 v138, v138, v140\n"
          "v_add_f32 v139, v139, v141\n"
          "s_nop 0\n"
          "v_lshlrev_b32 v140, 16, v91\n"
          "v_and_b32 v141, 0xffff0000, v91\n"
          "s_nop 0\n"
          "v_add_f32 v138, v138, v140\n"
          "v_add_f32 v139, v139, v141\n"
          "s_nop 0\n"
          "v_lshlrev_b32 v140, 16, v95\n"
          "v_and_b32 v141, 0xffff0000, v95\n"
          "s_nop 0\n"
          "v_add_f32 v138, v138, v140\n"
          "v_add_f32 v139, v139, v141\n"
          "s_nop 0\n"
          "v_lshlrev_b32 v140, 16, v99\n"
          "v_and_b32 v141, 0xffff0000, v99\n"
          "s_nop 0\n"
          "v_add_f32 v138, v138, v140\n"
          "v_add_f32 v139, v139, v141\n"
          "s_nop 0\n"
          ::: "v68","v69","v70","v71","v72","v73","v74","v75","v76","v77","v78","v79","v80","v81","v82","v83","v84","v85","v86","v87","v88","v89","v90","v91","v92","v93","v94","v95","v96","v97","v98","v99","v100","v101","v102","v103","v104","v105","v106","v107","v108","v109","v110","v111","v112","v113","v114","v115","v116","v117","v118","v119","v120","v121","v122","v123","v124","v125","v126","v127","v128","v129","v130","v131","v132","v133","v134","v135","v136","v137","v138","v139","v140","v141","v142","v143","v144","v145","v146","v147","v148","v149","v150","v151","v152", "memory");
          if (!LATELD && n + 2 < nch) loads0(n + 2);
        }
        const int qc = n & 1, np = (n >> 1) & 1, wr = (n >> 2) & 1, mt = (R.slice << lm) + (n >> 3);
        const unsigned soff = (unsigned)((rank * 1024 + row0 + wr * 128 + mt * 16) * 1024 + ((tw >> (8 * qc)) & 15) * 64 + np * 32) * 2u;
        asm volatile("v_mov_b32 v152, 0xc0135761\n"
          "v_mul_f32 v140, v132, v132\n"
          "v_mul_f32 v141, v133, v133\n"
          "s_nop 0\n"
          "v_mul_f32 v142, v134, v134\n"
          "v_mul_f32 v143, v135, v135\n"
          "s_nop 0\n"
          "v_mul_f32 v144, v136, v136\n"
          "v_mul_f32 v145, v137, v137\n"
          "s_nop 0\n"
          "v_mul_f32 v146, v138, v138\n"
          "v_mul_f32 v147, v139, v139\n"
          "s_nop 0\n"
          "v_fmamk_f32 v140, v140, 0xbdd2d3e7, v152\n"
          "v_fmamk_f32 v141, v141, 0xbdd2d3e7, v152\n"
          "s_nop 0\n"
          "v_fmamk_f32 v142, v142, 0xbdd2d3e7, v152\n"
          "v_fmamk_f32 v143, v143, 0xbdd2d3e7, v152\n"
          "s_nop 0\n"
          "v_fmamk_f32 v144, v144, 0xbdd2d3e7, v152\n"
          "v_fmamk_f32 v145, v145, 0xbdd2d3e7, v152\n"
          "s_nop 0\n"
          "v_fmamk_f32 v146, v146, 0xbdd2d3e7, v152\n"
          "v_fmamk_f32 v147, v147, 0xbdd2d3e7, v152\n"
          "s_nop 0\n"
          "v_mul_f32 v140, v132, v140\n"
          "v_mul_f32 v141, v133, v141\n"
          "s_nop 0\n"
          "v_mul_f32 v142, v134, v142\n"
          "v_mul_f32 v143, v135, v143\n"
          "s_nop 0\n"
          "v_mul_f32 v144, v136, v144\n"
          "v_mul_f32 v145, v137, v145\n"
          "s_nop 0\n"
          "v_mul_f32 v146, v138, v146\n"
          "v_mul_f32 v147, v139, v147\n"
          "s_nop 0\n"
          "v_exp_f32 v140, v140\n"
          "s_nop 0\n"
          "v_exp_f32 v141, v141\n"
          "s_nop 0\n"
          "v_exp_f32 v142, v142\n"
          "s_nop 0\n"
          "v_exp_f32 v143, v143\n"
          "s_nop 0\n"
          "v_exp_f32 v144, v144\n"
          "s_nop 0\n"
          "v_exp_f32 v145, v145\n"
          "s_nop 0\n"
          "v_exp_f32 v146, v146\n"
          "s_nop 0\n"
          "v_exp_f32 v147, v147\n"
          "s_nop 0\n"
          "v_add_f32 v140, 1.0, v140\n"
          "v_add_f32 v141, 1.0, v141\n"
          "s_nop 0\n"
          "v_add_f32 v142, 1.0, v142\n"
          "v_add_f32 v143, 1.0, v143\n"
          "s_nop 0\n"
          "v_add_f32 v144, 1.0, v144\n"
          "v_add_f32 v145, 1.0, v145\n"
          "s_nop 0\n"
          "v_add_f32 v146, 1.0, v146\n"
          "v_add_f32 v147, 1.0, v147\n"
          "s_nop 0\n"
          "v_rcp_f32 v140, v140\n"
          "s_nop 0\n"
          "v_rcp_f32 v141, v141\n"
          "s_nop 0\n"
          "v_rcp_f32 v142, v142\n"
          "s_nop 0\n"
          "v_rcp_f32 v143, v143\n"
          "s_nop 0\n"
          "v_rcp_f32 v144, v144\n"
          "s_nop 0\n"
          "v_rcp_f32 v145, v145\n"
          "s_nop 0\n"
          "v_rcp_f32 v146, v146\n"
          "s_nop 0\n"
          "v_rcp_f32 v147, v147\n"
          "s_nop 0\n"
          "v_mul_f32 v140, v132, v140\n"
          "v_mul_f32 v141, v133, v141\n"
          "s_nop 0\n"
          "v_mul_f32 v142, v134, v142\n"
          "v_mul_f32 v143, v135, v143\n"
          "s_nop 0\n"
          "v_mul_f32 v144, v136, v144\n"
          "v_mul_f32 v145, v137, v145\n"
          "s_nop 0\n"
          "v_mul_f32 v146, v138, v146\n"
          "v_mul_f32 v147, v139, v147\n"
          "s_nop 0\n"
          "v_cvt_pk_bf16_f32 v148, v140, v141\n"
          "v_cvt_pk_bf16_f32 v149, v142, v143\n"
          "s_nop 0\n"
          "v_cvt_pk_bf16_f32 v150, v144, v145\n"
          "v_cvt_pk_bf16_f32 v151, v146, v147\n"
          "s_nop 0\n"
          "s_nop 1\n"
          "v_permlane16_swap_b32 v148, v150\n"
          "v_permlane16_swap_b32 v149, v151\n"
          "s_nop 1\n"
          "v_or_b32 v148, %3, v148\n"
          "buffer_store_dwordx4 v[148:151], %0, %1, %2 offen sc0 sc1\n"
          "s_nop 1\n"
          :: "v"(voffO), "s"(qc ? srdO1 : srdO0), "s"(soff), "s"(badm) : "v68","v69","v70","v71","v72","v73","v74","v75","v76","v77","v78","v79","v80","v81","v82","v83","v84","v85","v86","v87","v88","v89","v90","v91","v92","v93","v94","v95","v96","v97","v98","v99","v100","v101","v102","v103","v104","v105","v106","v107","v108","v109","v110","v111","v112","v113","v114","v115","v116","v117","v118","v119","v120","v121","v122","v123","v124","v125","v126","v127","v128","v129","v130","v131","v132","v133","v134","v135","v136","v137","v138","v139","v140","v141","v142","v143","v144","v145","v146","v147","v148","v149","v150","v151","v152", "memory");
        if (LATELD && n + 2 < nch) { while (nb <= TOTAL && spare(LATELD)) __builtin_amdgcn_s_sleep(1); if (n & 1) loads1(n + 2); else loads0(n + 2); }
      }
      if (r == 0) STAMP(2); else STAMP(4);
    }
    wait_vm(0, 1700);
    asm volatile("s_waitcnt vmcnt(0)" ::: "memory");
    for (int r = 0; r + 1 < NR; ++r) {
      const Round R = round_cfg(p, r, b);
      for (int qc = 0; qc < 2; ++qc) { const int dest = (R.tw >> (16 * a + 8 * qc + 4)) & 7;
        if (lane == 0 && dest != rank) __hip_atomic_fetch_add(p.arr[dest], 32u >> R.ls, __ATOMIC_RELAXED, __HIP_MEMORY_SCOPE_SYSTEM); }
    }
    STAMP(7);
    while (nb <= TOTAL) tick();
    if (a == 0 && lane == 0) __hip_atomic_store(myws + W_CALL + 32 * b, call, __ATOMIC_RELAXED, __HIP_MEMORY_SCOPE_SYSTEM);
    if (a == 1 && b == 0) {
      const int src = (rank + 1 + lane % 7) & 7;
      if (wait_all_ge(myws + W_ARR + 32 * src, call * 2048u, myws + W_TMO, call) && lane == 0) p.out[rank][(size_t)src * 1024 * 1024] = 0x7fc0;
    }
    return;
  }
  __builtin_amdgcn_s_setprio(2);
  const int wr = wave >> 2, wc = wave & 3;
  unsigned tq = 0, call = 0;
  size_t outp = 0, arrp = 0;
  f32x32 acc0, acc1, acc2, acc3;
  int done = 0, nk_last = NKL;
  #pragma unroll 1
  for (int r = 0;; ++r) {
    const Round R = round_cfg(p, r, b);
    unsigned cnt = (ADAPT && r == NR - 1) ? nk_last : R.nk;
    const unsigned trig = r ? cnt - TRIG : 0xffffu;
    const unsigned sFlagOff = (W_FLAG + 16u * (((r - 1) * 256 + b) * 8 + wave)) * 4u;
    unsigned sB = (unsigned)(2 * done % 5) * 32768u, sA = (unsigned)((2 * done + 1) % 5) * 32768u, sT0, sT1;
    done += R.nk;
    unsigned vA0, vA1, vB0, vB1, vB2, vB3;
    { const unsigned lane = lane_id(), g = lane >> 4, fr = lane & 15, q = fr >> 2, pp = lane & 3;
      vA0 = ldsbase + sA + (wr*128 + fr)*128 + ((g ^ (fr>>1))*16); vA1 = vA0 ^ 64;
      const unsigned hB = (q>>1) | ((g&1)<<1), bB = ldsbase + sB + wc*8192 + (8*g+q)*128 + pp*8;
      vB0 = bB + ((0^hB)*32); vB1 = bB + ((1^hB)*32); vB2 = bB + ((2^hB)*32); vB3 = bB + ((3^hB)*32); }
    if (r == 0) { asm volatile("s_barrier" : "+v"(vA0), "+v"(vA1), "+v"(vB0), "+v"(vB1), "+v"(vB2), "+v"(vB3) :: "memory"); STAMP(1); }
    asm volatile(
      "ds_read_b128 v[128:131], %4 offset:0\n"
      "ds_read_b128 v[132:135], %4 offset:2048\n"
      "ds_read_b64_tr_b16 v[144:145], %6 offset:0\n"
      "ds_read_b64_tr_b16 v[146:147], %6 offset:512\n"
      "ds_read_b64_tr_b16 v[148:149], %7 offset:0\n"
      "ds_read_b64_tr_b16 v[150:151], %7 offset:512\n"
      "ds_read_b128 v[136:139], %4 offset:4096\n"
      "ds_read_b64_tr_b16 v[152:153], %8 offset:0\n"
      "ds_read_b64_tr_b16 v[154:155], %8 offset:512\n"
      "ds_read_b128 v[140:143], %4 offset:6144\n"
      "ds_read_b64_tr_b16 v[156:157], %9 offset:0\n"
      "ds_read_b64_tr_b16 v[158:159], %9 offset:512\n"
      "s_waitcnt lgkmcnt(0)\n"
      "s_waitcnt lgkmcnt(8)\n"
      "v_mfma_f32_16x16x32_bf16 v[0:3], v[144:147], v[128:131], 0\n"
      "v_mfma_f32_16x16x32_bf16 v[16:19], v[144:147], v[132:135], 0\n"
      "s_waitcnt lgkmcnt(6)\n"
      "v_mfma_f32_16x16x32_bf16 v[20:23], v[148:151], v[132:135], 0\n"
      "v_mfma_f32_16x16x32_bf16 v[4:7], v[148:151], v[128:131], 0\n"
      "s_setprio 2\n"
      "s_waitcnt lgkmcnt(5)\n"
      "v_mfma_f32_16x16x32_bf16 v[36:39], v[148:151], v[136:139], 0\n"
      "v_mfma_f32_16x16x32_bf16 v[32:35], v[144:147], v[136:139], 0\n"
      "s_waitcnt lgkmcnt(3)\n"
      "v_mfma_f32_16x16x32_bf16 v[40:43], v[152:155], v[136:139], 0\n"
      "v_mfma_f32_16x16x32_bf16 v[24:27], v[152:155], v[132:135], 0\n"
      "v_mfma_f32_16x16x32_bf16 v[8:11], v[152:155], v[128:131], 0\n"
      "s_waitcnt lgkmcnt(0)\n"
      "v_mfma_f32_16x16x32_bf16 v[12:15], v[156:159], v[128:131], 0\n"
      "ds_read_b128 v[128:131], %4 offset:8192\n"
      "v_mfma_f32_16x16x32_bf16 v[28:31], v[156:159], v[132:135], 0\n"
      "ds_read_b128 v[132:135], %4 offset:10240\n"
      "v_mfma_f32_16x16x32_bf16 v[44:47], v[156:159], v[136:139], 0\n"
      "ds_read_b128 v[136:139], %4 offset:12288\n"
      "v_mfma_f32_16x16x32_bf16 v[60:63], v[156:159], v[140:143], 0\n"
      "v_mfma_f32_16x16x32_bf16 v[56:59], v[152:155], v[140:143], 0\n"
      "v_mfma_f32_16x16x32_bf16 v[52:55], v[148:151], v[140:143], 0\n"
      "v_mfma_f32_16x16x32_bf16 v[48:51], v[144:147], v[140:143], 0\n"
      "ds_read_b128 v[140:143], %4 offset:14336\n"
      "s_waitcnt lgkmcnt(3)\n"
      "v_mfma_f32_16x16x32_bf16 v[64:67], v[144:147], v[128:131], 0\n"
      "v_mfma_f32_16x16x32_bf16 v[68:71], v[148:151], v[128:131], 0\n"
      "v_mfma_f32_16x16x32_bf16 v[72:75], v[152:155], v[128:131], 0\n"
      "v_mfma_f32_16x16x32_bf16 v[76:79], v[156:159], v[128:131], 0\n"
      "s_setprio 1\n"
      "ds_read_b128 v[128:131], %5 offset:0\n"
      "s_waitcnt lgkmcnt(3)\n"
      "v_mfma_f32_16x16x32_bf16 v[92:95], v[156:159], v[132:135], 0\n"
      "v_mfma_f32_16x16x32_bf16 v[88:91], v[152:155], v[132:135], 0\n"
      "v_mfma_f32_16x16x32_bf16 v[84:87], v[148:151], v[132:135], 0\n"
      "v_mfma_f32_16x16x32_bf16 v[80:83], v[144:147], v[132:135], 0\n"
      "ds_read_b128 v[132:135], %5 offset:2048\n"
      "s_waitcnt lgkmcnt(3)\n"
      "v_mfma_f32_16x16x32_bf16 v[96:99], v[144:147], v[136:139], 0\n"
      "s_waitcnt lgkmcnt(2)\n"
      "v_mfma_f32_16x16x32_bf16 v[112:115], v[144:147], v[140:143], 0\n"
      "ds_read_b64_tr_b16 v[144:145], %6 offset:4096\n"
      "ds_read_b64_tr_b16 v[146:147], %6 offset:4608\n"
      "v_mfma_f32_16x16x32_bf16 v[116:119], v[148:151], v[140:143], 0\n"
      "v_mfma_f32_16x16x32_bf16 v[100:103], v[148:151], v[136:139], 0\n"
      "ds_read_b64_tr_b16 v[148:149], %7 offset:4096\n"
      "ds_read_b64_tr_b16 v[150:151], %7 offset:4608\n"
      "v_mfma_f32_16x16x32_bf16 v[108:111], v[156:159], v[136:139], 0\n"
      "v_mfma_f32_16x16x32_bf16 v[104:107], v[152:155], v[136:139], 0\n"
      "ds_read_b128 v[136:139], %5 offset:4096\n"
      "v_mfma_f32_16x16x32_bf16 v[120:123], v[152:155], v[140:143], 0\n"
      "ds_read_b64_tr_b16 v[152:153], %8 offset:4096\n"
      "ds_read_b64_tr_b16 v[154:155], %8 offset:4608\n"
      "v_mfma_f32_16x16x32_bf16 v[124:127], v[156:159], v[140:143], 0\n"
      "ds_read_b128 v[140:143], %5 offset:6144\n"
      "ds_read_b64_tr_b16 v[156:157], %9 offset:4096\n"
      "ds_read_b64_tr_b16 v[158:159], %9 offset:4608\n"
      "s_branch 3f\n"
      "1:\n"
      "s_waitcnt lgkmcnt(8)\n"
      "v_mfma_f32_16x16x32_bf16 v[0:3], v[144:147], v[128:131], v[0:3]\n"
      "v_mfma_f32_16x16x32_bf16 v[16:19], v[144:147], v[132:135], v[16:19]\n"
      "s_waitcnt lgkmcnt(6)\n"
      "v_mfma_f32_16x16x32_bf16 v[20:23], v[148:151], v[132:135], v[20:23]\n"
      "v_mfma_f32_16x16x32_bf16 v[4:7], v[148:151], v[128:131], v[4:7]\n"
      "s_setprio 2\n"
      "s_waitcnt lgkmcnt(5)\n"
      "v_mfma_f32_16x16x32_bf16 v[36:39], v[148:151], v[136:139], v[36:39]\n"
      "v_mfma_f32_16x16x32_bf16 v[32:35], v[144:147], v[136:139], v[32:35]\n"
      "s_waitcnt lgkmcnt(3)\n"
      "v_mfma_f32_16x16x32_bf16 v[40:43], v[152:155], v[136:139], v[40:43]\n"
      "v_mfma_f32_16x16x32_bf16 v[24:27], v[152:155], v[132:135], v[24:27]\n"
      "v_mfma_f32_16x16x32_bf16 v[8:11], v[152:155], v[128:131], v[8:11]\n"
      "s_waitcnt lgkmcnt(0)\n"
      "v_mfma_f32_16x16x32_bf16 v[12:15], v[156:159], v[128:131], v[12:15]\n"
      "ds_read_b128 v[128:131], %4 offset:8192\n"
      "v_mfma_f32_16x16x32_bf16 v[28:31], v[156:159], v[132:135], v[28:31]\n"
      "ds_read_b128 v[132:135], %4 offset:10240\n"
      "v_mfma_f32_16x16x32_bf16 v[44:47], v[156:159], v[136:139], v[44:47]\n"
      "ds_read_b128 v[136:139], %4 offset:12288\n"
      "v_mfma_f32_16x16x32_bf16 v[60:63], v[156:159], v[140:143], v[60:63]\n"
      "v_mfma_f32_16x16x32_bf16 v[56:59], v[152:155], v[140:143], v[56:59]\n"
      "v_mfma_f32_16x16x32_bf16 v[52:55], v[148:151], v[140:143], v[52:55]\n"
      "v_mfma_f32_16x16x32_bf16 v[48:51], v[144:147], v[140:143], v[48:51]\n"
      "ds_read_b128 v[140:143], %4 offset:14336\n"
      "s_waitcnt lgkmcnt(3)\n"
      "v_mfma_f32_16x16x32_bf16 v[64:67], v[144:147], v[128:131], v[64:67]\n"
      "v_mfma_f32_16x16x32_bf16 v[68:71], v[148:151], v[128:131], v[68:71]\n"
      "v_mfma_f32_16x16x32_bf16 v[72:75], v[152:155], v[128:131], v[72:75]\n"
      "v_mfma_f32_16x16x32_bf16 v[76:79], v[156:159], v[128:131], v[76:79]\n"
      "s_setprio 1\n"
      "ds_read_b128 v[128:131], %5 offset:0\n"
      "s_waitcnt lgkmcnt(3)\n"
      "v_mfma_f32_16x16x32_bf16 v[92:95], v[156:159], v[132:135], v[92:95]\n"
      "v_mfma_f32_16x16x32_bf16 v[88:91], v[152:155], v[132:135], v[88:91]\n"
      "v_mfma_f32_16x16x32_bf16 v[84:87], v[148:151], v[132:135], v[84:87]\n"
      "v_mfma_f32_16x16x32_bf16 v[80:83], v[144:147], v[132:135], v[80:83]\n"
      "ds_read_b128 v[132:135], %5 offset:2048\n"
      "s_waitcnt lgkmcnt(3)\n"
      "v_mfma_f32_16x16x32_bf16 v[96:99], v[144:147], v[136:139], v[96:99]\n"
      "s_waitcnt lgkmcnt(2)\n"
      "v_mfma_f32_16x16x32_bf16 v[112:115], v[144:147], v[140:143], v[112:115]\n"
      "ds_read_b64_tr_b16 v[144:145], %6 offset:4096\n"
      "ds_read_b64_tr_b16 v[146:147], %6 offset:4608\n"
      "v_mfma_f32_16x16x32_bf16 v[116:119], v[148:151], v[140:143], v[116:119]\n"
      "v_mfma_f32_16x16x32_bf16 v[100:103], v[148:151], v[136:139], v[100:103]\n"
      "ds_read_b64_tr_b16 v[148:149], %7 offset:4096\n"
      "ds_read_b64_tr_b16 v[150:151], %7 offset:4608\n"
      "v_mfma_f32_16x16x32_bf16 v[108:111], v[156:159], v[136:139], v[108:111]\n"
      "v_mfma_f32_16x16x32_bf16 v[104:107], v[152:155], v[136:139], v[104:107]\n"
      "ds_read_b128 v[136:139], %5 offset:4096\n"
      "v_mfma_f32_16x16x32_bf16 v[120:123], v[152:155], v[140:143], v[120:123]\n"
      "ds_read_b64_tr_b16 v[152:153], %8 offset:4096\n"
      "ds_read_b64_tr_b16 v[154:155], %8 offset:4608\n"
      "v_mfma_f32_16x16x32_bf16 v[124:127], v[156:159], v[140:143], v[124:127]\n"
      "ds_read_b128 v[140:143], %5 offset:6144\n"
      "ds_read_b64_tr_b16 v[156:157], %9 offset:4096\n"
      "ds_read_b64_tr_b16 v[158:159], %9 offset:4608\n"
      "3:\n"
      "s_waitcnt lgkmcnt(8)\n"
      "v_mfma_f32_16x16x32_bf16 v[0:3], v[144:147], v[128:131], v[0:3]\n"
      "v_mfma_f32_16x16x32_bf16 v[16:19], v[144:147], v[132:135], v[16:19]\n"
      "s_waitcnt lgkmcnt(6)\n"
      "v_mfma_f32_16x16x32_bf16 v[20:23], v[148:151], v[132:135], v[20:23]\n"
      "s_add_u32 %10, %10, 65536\n"
      "s_mov_b32 %13, 65536\n"
      "v_mfma_f32_16x16x32_bf16 v[4:7], v[148:151], v[128:131], v[4:7]\n"
      "s_setprio 0\n"
      "s_waitcnt lgkmcnt(5)\n"
      "v_mfma_f32_16x16x32_bf16 v[36:39], v[148:151], v[136:139], v[36:39]\n"
      "s_cmp_ge_u32 %10, 163840\n"
      "s_cselect_b32 %13, 0xfffe8000, %13\n"
      "v_mfma_f32_16x16x32_bf16 v[32:35], v[144:147], v[136:139], v[32:35]\n"
      "s_waitcnt lgkmcnt(3)\n"
      "v_mfma_f32_16x16x32_bf16 v[40:43], v[152:155], v[136:139], v[40:43]\n"
      "s_cselect_b32 %14, 163840, 0\n"
      "s_sub_u32 %10, %10, %14\n"
      "v_mfma_f32_16x16x32_bf16 v[24:27], v[152:155], v[132:135], v[24:27]\n"
      "v_add_u32 %4, %13, %4\n"
      "v_mfma_f32_16x16x32_bf16 v[8:11], v[152:155], v[128:131], v[8:11]\n"
      "s_add_u32 %11, %11, 65536\n"
      "s_waitcnt lgkmcnt(0)\n"
      "v_mfma_f32_16x16x32_bf16 v[12:15], v[156:159], v[128:131], v[12:15]\n"
      "ds_read_b128 v[128:131], %5 offset:8192\n"
      "v_mfma_f32_16x16x32_bf16 v[28:31], v[156:159], v[132:135], v[28:31]\n"
      "s_cmp_ge_u32 %11, 163840\n"
      "s_cselect_b32 %14, 163840, 0\n"
      "ds_read_b128 v[132:135], %5 offset:10240\n"
      "v_mfma_f32_16x16x32_bf16 v[44:47], v[156:159], v[136:139], v[44:47]\n"
      "ds_read_b128 v[136:139], %5 offset:12288\n"
      "v_mfma_f32_16x16x32_bf16 v[60:63], v[156:159], v[140:143], v[60:63]\n"
      "s_sub_u32 %11, %11, %14\n"
      "s_sub_u32 %14, 65536, %14\n"
      "v_mfma_f32_16x16x32_bf16 v[56:59], v[152:155], v[140:143], v[56:59]\n"
      "v_add_u32 %6, %14, %6\n"
      "v_mfma_f32_16x16x32_bf16 v[52:55], v[148:151], v[140:143], v[52:55]\n"
      "v_add_u32 %7, %14, %7\n"
      "v_mfma_f32_16x16x32_bf16 v[48:51], v[144:147], v[140:143], v[48:51]\n"
      "v_add_u32 %8, %14, %8\n"
      "ds_read_b128 v[140:143], %5 offset:14336\n"
      "s_waitcnt lgkmcnt(3)\n"
      "v_mfma_f32_16x16x32_bf16 v[64:67], v[144:147], v[128:131], v[64:67]\n"
      "v_add_u32 %9, %14, %9\n"
      "v_mfma_f32_16x16x32_bf16 v[68:71], v[148:151], v[128:131], v[68:71]\n"
      "v_add_u32 %5, %13, %5\n"
      "v_mfma_f32_16x16x32_bf16 v[72:75], v[152:155], v[128:131], v[72:75]\n"
      "v_mfma_f32_16x16x32_bf16 v[76:79], v[156:159], v[128:131], v[76:79]\n"
      "s_waitcnt lgkmcnt(0)\n"
      "s_barrier\n"
      "s_setprio 3\n"
      "ds_read_b128 v[128:131], %4 offset:0\n"
      "v_mfma_f32_16x16x32_bf16 v[92:95], v[156:159], v[132:135], v[92:95]\n"
      "v_mfma_f32_16x16x32_bf16 v[88:91], v[152:155], v[132:135], v[88:91]\n"
      "v_mfma_f32_16x16x32_bf16 v[84:87], v[148:151], v[132:135], v[84:87]\n"
      "v_mfma_f32_16x16x32_bf16 v[80:83], v[144:147], v[132:135], v[80:83]\n"
      "ds_read_b128 v[132:135], %4 offset:2048\n"
      "v_mfma_f32_16x16x32_bf16 v[96:99], v[144:147], v[136:139], v[96:99]\n"
      "s_cmp_lg_u32 %12, %15\n"
      "v_mfma_f32_16x16x32_bf16 v[112:115], v[144:147], v[140:143], v[112:115]\n"
      "s_cbranch_scc1 2f\n"
      "s_waitcnt vmcnt(0)\n"
      "v_mov_b32 v160, %16\n"
      "v_mov_b32 v161, %17\n"
      "global_store_dword v160, v161, %18 sc0 sc1\n"
      "2:\n"
      "ds_read_b64_tr_b16 v[144:145], %6 offset:0\n"
      "ds_read_b64_tr_b16 v[146:147], %6 offset:512\n"
      "v_mfma_f32_16x16x32_bf16 v[116:119], v[148:151], v[140:143], v[116:119]\n"
      "v_mfma_f32_16x16x32_bf16 v[100:103], v[148:151], v[136:139], v[100:103]\n"
      "ds_read_b64_tr_b16 v[148:149], %7 offset:0\n"
      "ds_read_b64_tr_b16 v[150:151], %7 offset:512\n"
      "v_mfma_f32_16x16x32_bf16 v[108:111], v[156:159], v[136:139], v[108:111]\n"
      "v_mfma_f32_16x16x32_bf16 v[104:107], v[152:155], v[136:139], v[104:107]\n"
      "s_sub_u32 %12, %12, 1\n"
      "ds_read_b128 v[136:139], %4 offset:4096\n"
      "v_mfma_f32_16x16x32_bf16 v[120:123], v[152:155], v[140:143], v[120:123]\n"
      "s_cmp_lg_u32 %12, 0\n"
      "ds_read_b64_tr_b16 v[152:153], %8 offset:0\n"
      "ds_read_b64_tr_b16 v[154:155], %8 offset:512\n"
      "v_mfma_f32_16x16x32_bf16 v[124:127], v[156:159], v[140:143], v[124:127]\n"
      "ds_read_b128 v[140:143], %4 offset:6144\n"
      "ds_read_b64_tr_b16 v[156:157], %9 offset:0\n"
      "ds_read_b64_tr_b16 v[158:159], %9 offset:512\n"
      "s_cbranch_scc1 1b\n"
      "s_waitcnt lgkmcnt(0)\n"
      "s_nop 7\n"
      : "={v[0:31]}"(acc0), "={v[32:63]}"(acc1), "={v[64:95]}"(acc2), "={v[96:127]}"(acc3),
        "+{v162}"(vA0), "+{v163}"(vA1), "+{v164}"(vB0), "+{v165}"(vB1), "+{v166}"(vB2), "+{v167}"(vB3), "+s"(sA), "+s"(sB), "+s"(cnt), "=&s"(sT0), "=&s"(sT1)
      : "s"(trig), "s"(sFlagOff), "s"(call), "s"(myws)
      : "v128","v129","v130","v131","v132","v133","v134","v135","v136","v137","v138","v139","v140","v141","v142","v143","v144","v145","v146","v147","v148","v149","v150","v151","v152","v153","v154","v155","v156","v157","v158","v159","v160","v161", "scc", "memory");
    if (r == 0) { int bb = b; asm volatile("" : "+s"(bb)); call = load_call(); tq = (round_cfg(p, NR - 1, bb).tw >> (8 * wc)) & 127; }
    if (r == NR - 2) { unsigned i = tq >> 4; asm volatile("" : "+s"(i)); outp = (size_t)p.out[i & 7]; arrp = (size_t)p.arr[i & 7]; }
    if (ADAPT) nk_last = rng_nk(sload_now(myws + W_RNG + 16 * b));
    if (LDELAY && r + 1 == NR && (int)(tq >> 4) == rank) for (int i = 0; i < LDELAY; ++i) __builtin_amdgcn_s_sleep(8);
    const unsigned lo = lane_id() * 16u;
    const unsigned so = (r * RSTR + b) * PBLK + wave * PWAVE, rx = r + 1 == NR ? (unsigned)(R.slice << LM) * PMT : 0u;
    dump_mt<0>(acc1, lo, srdP, so + ((2 * PMT) ^ rx)); dump_mt<1>(acc1, lo, srdP, so + ((3 * PMT) ^ rx));
    dump_mt<0>(acc2, lo, srdP, so + ((4 * PMT) ^ rx)); dump_mt<1>(acc2, lo, srdP, so + ((5 * PMT) ^ rx));
    dump_mt<0>(acc3, lo, srdP, so + ((6 * PMT) ^ rx)); dump_mt<1>(acc3, lo, srdP, so + ((7 * PMT) ^ rx));
    if (LM == 0) dump_mt<1>(acc0, lo, srdP, so + (PMT ^ rx));
    if (r + 1 == NR) break;
    dump_mt<0>(acc0, lo, srdP, so);
    if (LM == 1) dump_mt<1>(acc0, lo, srdP, so + PMT);
    asm volatile("" : "+s"(outp), "+s"(arrp), "+s"(tq), "+s"(call));
  }
  { constexpr int r = NR - 1;
    const Round R = round_cfg(p, r, b);
    const unsigned ln = lane_id();
    const unsigned tmo0 = __hip_atomic_load(myws + W_TMO, __ATOMIC_RELAXED, __HIP_MEMORY_SCOPE_AGENT);
    asm volatile("s_waitcnt vmcnt(0)" ::: "memory");
    if (ln == 0) __hip_atomic_store(myws + W_FLAG + 16 * ((r * 256 + b) * 8 + wave), call, __ATOMIC_RELAXED, __HIP_MEMORY_SCOPE_SYSTEM);
    STAMP(11);
    bool bad = false;
    { unsigned* q = myws + W_FLAG + 16 * ((r * 256 + part_blk(R, ln & (SLAST - 1))) * 8 + wave);
      const unsigned long long t0 = __builtin_amdgcn_s_memrealtime();
      bool ok = false;
      for (unsigned spins = 0;; ++spins) {
        if (!ok) ok = (int)(__hip_atomic_load(q, __ATOMIC_RELAXED, __HIP_MEMORY_SCOPE_SYSTEM) - call) >= 0;
        if (__all(ok)) break;
        if ((spins & 63) == 63 && __any(past_deadline(t0, myws + W_TMO, call))) { bad = true; break; }
        __builtin_amdgcn_s_sleep(PSLEEP);
      }
      bad |= tmo0 == call; }
    asm volatile("" ::: "memory");
    STAMP(12);
    const __amdgpu_buffer_rsrc_t srdO = __builtin_amdgcn_make_buffer_rsrc((void*)outp, (short)0, 8192 * 1024 * 2, 0x00020000);
    unsigned pso[3];
    #pragma unroll
    for (int j = 0; j < 3; ++j) pso[j] = (r * RSTR + part_blk(R, R.slice ^ (j + 1))) * PBLK + wave * PWAVE + R.slice * 2 * PMT;
    final_slice(acc0, pso, ln * 16u, srdP, (ln & 15) * 2048u + (ln & 16) * 2u + (ln & 32) / 2u, srdO,
                (unsigned)((rank * 1024 + row0 + wr * 128 + R.slice * 32) * 1024 + (tq & 15) * 64) * 2u, bad, (int)(tq >> 4) != rank);
    asm volatile("s_waitcnt vmcnt(0)" ::: "memory");
    if (ln == 0 && (int)(tq >> 4) != rank) __hip_atomic_fetch_add((__attribute__((address_space(1))) unsigned*)arrp, 16u / SLAST, __ATOMIC_RELAXED, __HIP_MEMORY_SCOPE_SYSTEM);
  }
}

extern "C" void kernel_launch(int rank, int nranks, ncclComm_t comm, hipStream_t stream,
                              void* const* d_in, const int* in_sizes, int n_in,
                              void* d_out, int out_size, const int* in_dtypes, int out_dtype,
                              void* const* d_out_all, void* const* d_ws_all, size_t ws_bytes) {
  if (nranks != 8 || (unsigned)rank >= 8u || ws_bytes < PART_OFF + (size_t)NR * 256 * PBLK) return;
  (void)hipFuncSetAttribute((const void*)gemm_a2a, hipFuncAttributeMaxDynamicSharedMemorySize, LDS_BYTES);
  Params p;
  p.x = (const char*)d_in[0]; p.w = (const char*)d_in[1]; p.myws = (unsigned*)d_ws_all[rank];
  for (int i = 0; i < 8; ++i) { p.out[i] = (unsigned short*)d_out_all[i]; p.arr[i] = (unsigned*)d_ws_all[i] + W_ARR + 32 * rank; }
  p.rank = rank; p.pad = 0;
  for (int i = 0; i < 128; ++i) { unsigned w = 0; for (int q = 0; q < 4; ++q) { const unsigned t = TILES[i] >> (8 * q); w |= (((rank + t) & 7) * 16 + ((t >> 3) & 15)) << (8 * q); } p.tiles[i] = w; }
  gemm_a2a<<<256, 768, LDS_BYTES, stream>>>(p);
}
```
